```python
import numpy as np

kernel_name = "gpu_gemm_ar_m4096_k4096_n16384_e4m3e4m3_dequant"


ABSMAX_RTOL = 0.03

COMPUTE_DTYPE = "float32"
EPILOGUE = "dequant"
IN_QUANT = None
OUT_QUANT = None
IN_FP8 = "e4m3e4m3"
IN_DTYPES = {"x": "e4m3", "w_mat": "e4m3",
             "scale_x": "float32", "scale_w": "float32"}
WORLD_SIZE = 8

def _snap_fp8(x, mant_bits, max_abs, min_normal):
    a = np.abs(x.astype(np.float64))
    _m, _e = np.frexp(a)
    q = np.where(a >= min_normal, np.ldexp(1.0, _e - 1 - mant_bits),
                 min_normal * 2.0 ** -mant_bits)
    snapped = np.minimum(np.round(a / q) * q, max_abs)
    return (np.sign(x) * snapped).astype(np.float32)

def setup_inputs():

    x = _snap_fp8(np.random.randn(4096, 4096).astype(np.float32), 3, 448.0, 2.0 ** -6)
    w_mat = _snap_fp8(np.random.randn(4096, 16384).astype(np.float32), 3, 448.0, 2.0 ** -6)
    scale_x = (0.02 * (1.0 + np.random.rand(1))).astype(np.float32)
    scale_w = (0.02 * (1.0 + np.random.rand(1))).astype(np.float32)
    return {"x": x, "w_mat": w_mat, "scale_x": scale_x, "scale_w": scale_w}

def reference(x, w_mat, scale_x, scale_w):
    acc = (x @ w_mat).astype(np.float32)
    y = (acc * (scale_x * scale_w)).astype(np.float32)
    return y
```

```cpp
#include <hip/hip_runtime.h>
#include <rccl/rccl.h>
#include <cstdio>

#define HIP_CHECK(expr) do { hipError_t _e = (expr); if (_e != hipSuccess) { \
    fprintf(stderr, "HIP error %s at %s:%d\n", hipGetErrorString(_e), __FILE__, __LINE__); \
    return; } } while (0)

namespace {

#define LAS __attribute__((address_space(3)))
typedef int i32x4 __attribute__((ext_vector_type(4)));
typedef int i32x8 __attribute__((ext_vector_type(8)));
typedef float f32x4 __attribute__((ext_vector_type(4)));
typedef unsigned u32x2 __attribute__((ext_vector_type(2)));
typedef unsigned u32x4 __attribute__((ext_vector_type(4)));
typedef unsigned long long u64;

constexpr int NR = 8;
constexpr int M = 4096, N = 16384, KL = 512;
constexpr int NWG = 256, NPUSH = 16, PUSH_WAVES = 4;
constexpr int NBLK = 80;
constexpr int NKS = 32;
constexpr unsigned PIECE = 4096;
constexpr unsigned KSB = 8 * PIECE;
constexpr unsigned BLKB = NKS * KSB;
constexpr unsigned HTB = 16384;
constexpr int NJOBS = NBLK * 32;

constexpr size_t OFF_CNT = 0;
constexpr size_t OFF_TMO = 32768;
constexpr size_t OFF_FLAG = 65536;
constexpr unsigned FLAGB = NBLK * 256;
constexpr size_t OFF_DATA = 1u << 20;
constexpr unsigned DATAB = NBLK * BLKB;
constexpr size_t WS_NEED = OFF_DATA + 2 * (size_t)DATAB;
constexpr u64 DEADLINE_TICKS = 50000000ull;

struct Params {
    const unsigned char* x; const unsigned char* w; const float* sx; const float* sw;
    float* out; unsigned char* ws[NR]; int rank;
};

#define RSRC(ptr, bytes) __builtin_amdgcn_make_buffer_rsrc((void*)(ptr), 0, (unsigned)(bytes), 0x00020000)
#define WAIT_V(n) asm volatile("s_waitcnt vmcnt(" #n ")" ::: "memory")
#define WAIT_L(n) asm volatile("s_waitcnt lgkmcnt(" #n ")" ::: "memory")
#define BAR __builtin_amdgcn_s_barrier()
#define SCHED __builtin_amdgcn_sched_barrier(0)
constexpr int SYS = 17;
#define STAMP(i) do {} while (0)

__device__ __forceinline__ void job_decode(int n, int& blk, int& c) {
    if (n < 992) { c = n / 31; blk = n - c * 31; }
    else if (n < 2432) { const int q = n - 992, r = q / 480, rem = q - r * 480; c = rem / 15; blk = 31 + 15 * r + (rem - c * 15); }
    else { const int q = n - 2432; c = q >> 2; blk = 76 + (q & 3); }
}

__device__ __forceinline__ void job_load(const Params& p, int blk, int c, int lane, unsigned (&r)[16]) {
    if (blk < 16) {
        const auto rs = RSRC(p.x, (unsigned)M * KL);
#pragma unroll
        for (int i = 0; i < 4; ++i) {
            const u32x4 v = __builtin_amdgcn_raw_buffer_load_b128(rs, (unsigned)(lane + 64 * i) * KL, (unsigned)(blk * 256 * KL + c * 16), 0);
            r[4 * i] = v[0]; r[4 * i + 1] = v[1]; r[4 * i + 2] = v[2]; r[4 * i + 3] = v[3];
        }
    } else {
        const auto rs = RSRC(p.w, (unsigned)KL * N);
#pragma unroll
        for (int k = 0; k < 16; ++k)
            r[k] = __builtin_amdgcn_raw_buffer_load_b32(rs, (unsigned)lane * 4u, (unsigned)((c * 16 + k) * N + (blk - 16) * 256), 0);
    }
}
__device__ __forceinline__ void job_pack(int blk, const unsigned (&r)[16], u32x4 (&d)[4]) {
    if (blk < 16) {
#pragma unroll
        for (int i = 0; i < 4; ++i) d[i] = (u32x4){r[4 * i], r[4 * i + 1], r[4 * i + 2], r[4 * i + 3]};
    } else {
#pragma unroll
        for (int q = 0; q < 4; ++q) {
            const unsigned t0 = __builtin_amdgcn_perm(r[4 * q + 1], r[4 * q], 0x05010400u), t1 = __builtin_amdgcn_perm(r[4 * q + 1], r[4 * q], 0x07030602u);
            const unsigned t2 = __builtin_amdgcn_perm(r[4 * q + 3], r[4 * q + 2], 0x05010400u), t3 = __builtin_amdgcn_perm(r[4 * q + 3], r[4 * q + 2], 0x07030602u);
            d[0][q] = __builtin_amdgcn_perm(t2, t0, 0x05040100u); d[1][q] = __builtin_amdgcn_perm(t2, t0, 0x07060302u);
            d[2][q] = __builtin_amdgcn_perm(t3, t1, 0x05040100u); d[3][q] = __builtin_amdgcn_perm(t3, t1, 0x07060302u);
        }
    }
}

__device__ __forceinline__ bool tile_of(int r, int b, int& mb, int& nb) {
    const int x = b & 7, idx = (b >> 3) - 2, s = x >> 1, h = x & 1;
    if (r < 4) { const int L = h * 30 + idx; mb = 4 * s + (L & 3); nb = 15 * r + (L >> 2); return true; }
    if (r == 4 && idx < 8) { mb = 4 * s + (idx & 3); nb = 60 + 2 * h + (idx >> 2); return true; }
    return false;
}

__global__ void __launch_bounds__(512, 2) gemm_ar_kernel(Params p) {
    __shared__ __attribute__((aligned(16))) unsigned char smem[8 * HTB];
    LAS unsigned char* lds = (LAS unsigned char*)smem;
    const int tid = threadIdx.x, wid = __builtin_amdgcn_readfirstlane(tid >> 6), lane = tid & 63;
    const int b = blockIdx.x;
    unsigned char* const me = p.ws[p.rank];
    unsigned* const cntp = (unsigned*)(me + OFF_CNT + (size_t)b * 128);
    unsigned* const tmop = (unsigned*)(me + OFF_TMO);
    const unsigned call = __builtin_amdgcn_readfirstlane(__hip_atomic_load(cntp, __ATOMIC_RELAXED, __HIP_MEMORY_SCOPE_AGENT)) + 1u;
    const unsigned par = call & 1u, tag = ((call + 1u) >> 1) & 0xffu;
    STAMP(0);

    if (b < NPUSH) {
        if (wid >= PUSH_WAVES) return;
        constexpr int NW = NPUSH * PUSH_WAVES;
        const int v = b * PUSH_WAVES + wid;
        unsigned char* dbase[NR - 1];
#pragma unroll
        for (int q = 0; q < NR - 1; ++q) {
            const int peer = (p.rank + 1 + (q + v) % (NR - 1)) & (NR - 1), rel = (p.rank - peer - 1) & (NR - 1);
            dbase[q] = p.ws[peer] + OFF_DATA + (size_t)par * DATAB + (size_t)rel * PIECE;
        }
        unsigned char* fptr;
        { const int q = lane < NR - 1 ? lane : 0, peer = (p.rank + 1 + q) & (NR - 1), rel = (p.rank - peer - 1) & (NR - 1);
          fptr = p.ws[peer] + OFF_FLAG + (size_t)par * FLAGB + rel; }
        const unsigned jl = (unsigned)lane * 16u;
        unsigned r[16]; u32x4 d[4];
        int blk, c; job_decode(v, blk, c);
        job_load(p, blk, c, lane, r);
        int prev = -1;
        for (int n = v; n < NJOBS; n += NW) {
            job_pack(blk, r, d);
            const int pi = blk * 256 + 32 + 7 * c;
            if (n + NW < NJOBS) { job_decode(n + NW, blk, c); job_load(p, blk, c, lane, r); }
#pragma unroll
            for (int i = 0; i < 4; ++i) {
#pragma unroll
                for (int q = 0; q < NR - 1; ++q)
                    __builtin_amdgcn_raw_buffer_store_b128(d[i], RSRC(dbase[q], DATAB - (NR - 2) * PIECE), jl + i * 1024u, (unsigned)pi * PIECE, SYS);
                WAIT_V(7);
                if (i == 0 && prev >= 0 && lane < NR - 1) __hip_atomic_store(fptr + prev, (unsigned char)tag, __ATOMIC_RELAXED, __HIP_MEMORY_SCOPE_SYSTEM);
            }
            prev = pi;
        }
        WAIT_V(0);
        if (lane < NR - 1) __hip_atomic_store(fptr + prev, (unsigned char)tag, __ATOMIC_RELAXED, __HIP_MEMORY_SCOPE_SYSTEM);
        if (tid == 0) __hip_atomic_store(cntp, call, __ATOMIC_RELAXED, __HIP_MEMORY_SCOPE_AGENT);
        STAMP(1);
        return;
    }

    const auto rsD = RSRC(me + OFF_DATA + (size_t)par * DATAB, DATAB);
    const auto rsF = RSRC(me + OFF_FLAG + (size_t)par * FLAGB, FLAGB);
    {
        constexpr int NW = (NWG - NPUSH) * 8;
        const int v = (b - NPUSH) + (NWG - NPUSH) * wid;
        unsigned r0[16], r1[16]; u32x4 d[4];
        int blk0, c0, blk1 = 0, c1 = 0;
        job_decode(v, blk0, c0); job_load(p, blk0, c0, lane, r0);
        const bool two = v + NW < NJOBS;
        if (two) { job_decode(v + NW, blk1, c1); job_load(p, blk1, c1, lane, r1); }
        job_pack(blk0, r0, d);
#pragma unroll
        for (int i = 0; i < 4; ++i) __builtin_amdgcn_raw_buffer_store_b128(d[i], rsD, (unsigned)lane * 16u + i * 1024u, (unsigned)(blk0 * 256 + c0) * PIECE, SYS);
        if (two) {
            u32x4 e[4]; job_pack(blk1, r1, e);
#pragma unroll
            for (int i = 0; i < 4; ++i) __builtin_amdgcn_raw_buffer_store_b128(e[i], rsD, (unsigned)lane * 16u + i * 1024u, (unsigned)(blk1 * 256 + c1) * PIECE, SYS);
        }
        WAIT_V(0);
        if (lane == 0 || (lane == 1 && two))
            __builtin_amdgcn_raw_buffer_store_b8((unsigned char)tag, rsF, (unsigned)(lane ? blk1 * 256 + c1 : blk0 * 256 + c0), 0, SYS);
    }

    STAMP(1);
    const int wr = wid >> 2, wc = wid & 3;
    const float scale = p.sx[0] * p.sw[0];
    const unsigned tag4 = tag * 0x01010101u;
    const unsigned voff = (unsigned)(wid * PIECE + lane * 16), ldsw = (unsigned)wid * 2048u;
    unsigned aoff = (unsigned)(size_t)lds + (lane >> 4) * 2048 + (wr * 64 + (lane & 15)) * 16;
    unsigned boff = (unsigned)(size_t)lds + 4 * HTB + (lane >> 4) * 2048 + (wc * 16 + (lane & 15)) * 16;
    asm volatile("" : "+v"(aoff), "+v"(boff));
#define SA(bf, h) (((bf) * 2 + (h)) * HTB)
#define SB(bf, h) ((4 + (bf) * 2 + (h)) * HTB)
#define STAGE(bufoff, soff) do { \
        __builtin_amdgcn_raw_ptr_buffer_load_lds(rsD, (LAS void*)(lds + (bufoff) + ldsw), 16, voff, (soff), 0, SYS); \
        __builtin_amdgcn_raw_ptr_buffer_load_lds(rsD, (LAS void*)(lds + (bufoff) + ldsw), 16, voff, (soff), 1024, SYS); } while (0)
#define RD(base, off) (*(const LAS i32x4*)(size_t)((base) + (off)))
#define LDA(dst, bf, h) do { _Pragma("unroll") for (int m = 0; m < 4; ++m) { \
        const i32x4 lo = RD(aoff, SA(bf, h) + m * 256), hi = RD(aoff, SA(bf, h) + m * 256 + 8192); \
        dst[m] = __builtin_shufflevector(lo, hi, 0, 1, 2, 3, 4, 5, 6, 7); } } while (0)
#define LDB(dst, bf, h) do { _Pragma("unroll") for (int n = 0; n < 2; ++n) { \
        const i32x4 lo = RD(boff, SA(bf, h) + n * 1024), hi = RD(boff, SA(bf, h) + n * 1024 + 8192); \
        dst[n] = __builtin_shufflevector(lo, hi, 0, 1, 2, 3, 4, 5, 6, 7); } } while (0)
#define MMA(ai, bj, At, Bt) do { __builtin_amdgcn_s_setprio(1); \
        _Pragma("unroll") for (int m = 0; m < 4; ++m) _Pragma("unroll") for (int n = 0; n < 2; ++n) \
            asm volatile("v_mfma_f32_16x16x128_f8f6f4 %0, %1, %2, %0" : "+v"(acc[ai][bj][m][n]) : "v"(At[m]), "v"(Bt[n])); \
        __builtin_amdgcn_s_setprio(0); } while (0)

    u64 sm = 0, nm = 0;
    unsigned sf, nf = 0;
    u32x2 fl = {0u, 0u};
    int fl_tgt = 0;
    unsigned tv = 0;
#define FLAG_OFF(mb_, nb_) ((unsigned)((lane < 32 ? (mb_) : 16 + (nb_)) * 256 + (lane & 31) * 8))
#define READY(f) __builtin_amdgcn_ballot_w64((f)[0] == tag4 && (f)[1] == tag4)
#define NEED(ks) do { const u64 need_ = 0x100000001ull << (ks); \
        if ((sm & need_) != need_) { \
            const u64 t0_ = __builtin_amdgcn_s_memrealtime(); \
            for (;;) { u32x2 f_; unsigned t_; \
                asm volatile("buffer_load_dwordx2 %0, %2, %3, 0 offen sc0 sc1\n\tglobal_load_dword %1, %4, %5 sc1\n\ts_waitcnt vmcnt(0)" \
                             : "=&v"(f_), "=&v"(t_) : "v"(sf), "s"(rsF), "v"(0u), "s"(tmop) : "memory"); \
                sm = READY(f_); \
                if ((sm & need_) == need_) break; \
                if (__builtin_amdgcn_readfirstlane(t_) == call || __builtin_amdgcn_s_memrealtime() - t0_ > DEADLINE_TICKS) { \
                    if (lane == 0) __hip_atomic_store(tmop, call, __ATOMIC_RELAXED, __HIP_MEMORY_SCOPE_AGENT); \
                    sm = ~0ull; break; } \
                __builtin_amdgcn_s_sleep(1); } \
        } } while (0)
#define REFRESH() do { asm volatile("" : "+v"(fl)); \
        if (fl_tgt == 1) sm |= READY(fl); else if (fl_tgt == 2) nm |= READY(fl); \
        fl_tgt = sm != ~0ull ? 1 : 2; \
        const unsigned fo_ = fl_tgt == 1 ? sf : nf; \
        asm volatile("buffer_load_dwordx2 %0, %2, %3, 0 offen sc0 sc1\n\tglobal_load_dword %1, %4, %5 sc1" \
                     : "+v"(fl), "+v"(tv) : "v"(fo_), "s"(rsF), "v"(0u), "s"(tmop) : "memory"); } while (0)

    int mb, nb; tile_of(0, b, mb, nb);
    f32x4 acc[2][2][4][2];
#pragma unroll
    for (int a = 0; a < 2; ++a)
#pragma unroll
        for (int c = 0; c < 2; ++c)
#pragma unroll
            for (int m = 0; m < 4; ++m)
#pragma unroll
                for (int n = 0; n < 2; ++n) acc[a][c][m][n] = (f32x4){0.f, 0.f, 0.f, 0.f};
    i32x8 At[4], B0[2], B1[2];
    unsigned cA = (unsigned)mb * BLKB, cB = (unsigned)(16 + nb) * BLKB;
    constexpr unsigned HS = 2048;
    sf = FLAG_OFF(mb, nb);
    NEED(0); NEED(1);
    STAGE(SB(0, 0), cB); STAGE(SA(0, 0), cA); STAGE(SB(0, 1), cB + HS); STAGE(SA(0, 1), cA + HS);
    if (wr == 1) BAR;
    WAIT_V(4); BAR;
    STAGE(SB(1, 0), cB + KSB); STAGE(SA(1, 0), cA + KSB); STAGE(SB(1, 1), cB + HS + KSB);
    WAIT_V(6); BAR;
    STAMP(2);
    for (int r = 0;; ++r) {
        int nmb = mb, nnb = nb;
        const bool has_next = tile_of(r + 1, b, nmb, nnb);
        const unsigned nA = (unsigned)nmb * BLKB, nB = (unsigned)(16 + nnb) * BLKB;
        nf = FLAG_OFF(nmb, nnb); nm = 0;
        if (fl_tgt == 2) fl_tgt = 0;
        for (int t = 0; t < NKS; t += 2) {
            const bool last = (t == NKS - 2);
            const unsigned a1 = cA + (unsigned)(t + 1) * KSB;
            const unsigned a2 = last ? nA : cA + (unsigned)(t + 2) * KSB, b2 = last ? nB : cB + (unsigned)(t + 2) * KSB;
            const unsigned a3 = a2 + KSB, b3 = b2 + KSB;
            const int k2 = last ? 0 : t + 2;
            if (last) { sm = nm; sf = nf; fl_tgt = fl_tgt == 2 ? 1 : 0; }
            LDB(B0, 0, 0); SCHED; LDA(At, 0, 0); STAGE(SA(1, 1), a1 + HS);
            WAIT_L(8); BAR; WAIT_L(0); SCHED; MMA(0, 0, At, B0); BAR; SCHED;
            LDB(B1, 0, 1); SCHED; NEED(k2); STAGE(SB(0, 0), b2);
            BAR; WAIT_L(0); SCHED; MMA(0, 1, At, B1); BAR; SCHED;
            LDA(At, 0, 1); STAGE(SA(0, 0), a2);
            BAR; WAIT_L(0); SCHED; MMA(1, 0, At, B0); BAR; SCHED;
            STAGE(SB(0, 1), b2 + HS);
            WAIT_V(6); SCHED; REFRESH(); SCHED; BAR; SCHED; MMA(1, 1, At, B1); BAR; SCHED;
            LDB(B0, 1, 0); SCHED; LDA(At, 1, 0); STAGE(SA(0, 1), a2 + HS);
            WAIT_L(8); BAR; WAIT_L(0); SCHED; MMA(0, 0, At, B0); BAR; SCHED;
            LDB(B1, 1, 1); SCHED; NEED(k2 + 1); STAGE(SB(1, 0), b3);
            BAR; WAIT_L(0); SCHED; MMA(0, 1, At, B1); BAR; SCHED;
            LDA(At, 1, 1); STAGE(SA(1, 0), a3);
            BAR; WAIT_L(0); SCHED; MMA(1, 0, At, B0); BAR; SCHED;
            STAGE(SB(1, 1), b3 + HS);
            WAIT_V(6); BAR; SCHED; MMA(1, 1, At, B1); BAR; SCHED;
        }
        asm volatile("s_nop 15\n\ts_nop 7" : "+v"(tv) :: "memory"); SCHED;
        {
            const float sc = __builtin_amdgcn_readfirstlane(tv) == call ? __builtin_nanf("") : scale;
            const auto rs = RSRC(p.out + (size_t)mb * 256 * N + nb * 256, 256u * N * 4u);
            int tz = threadIdx.x; asm volatile("" : "+v"(tz));
            const unsigned o0 = (unsigned)((wr * 64 + 4 * ((tz >> 4) & 3)) * N + wc * 64 + 4 * (tz & 15)) * 4u;
#pragma unroll
            for (int ai = 0; ai < 2; ++ai)
#pragma unroll
                for (int m = 0; m < 4; ++m)
#pragma unroll
                    for (int t = 0; t < 4; ++t) {
                        const f32x4 v = {acc[ai][0][m][0][t] * sc, acc[ai][0][m][1][t] * sc, acc[ai][1][m][0][t] * sc, acc[ai][1][m][1][t] * sc};
                        __builtin_amdgcn_raw_buffer_store_b128(__builtin_bit_cast(u32x4, v), rs, o0 + (unsigned)((ai * 128 + m * 16 + t) * N) * 4u, 0, 2);
                    }
        }
        STAMP(3 + r);
        if (!has_next) break;
#pragma unroll
        for (int a = 0; a < 2; ++a)
#pragma unroll
            for (int c = 0; c < 2; ++c)
#pragma unroll
                for (int m = 0; m < 4; ++m)
#pragma unroll
                    for (int n = 0; n < 2; ++n) { acc[a][c][m][n] = (f32x4){0.f, 0.f, 0.f, 0.f}; asm volatile("" : "+v"(acc[a][c][m][n])); }
        SCHED;
        mb = nmb; nb = nnb; cA = nA; cB = nB;
    }
    WAIT_V(0);
    if (wr == 0) BAR;
    STAMP(15);
    if (tid == 0) __hip_atomic_store(cntp, call, __ATOMIC_RELAXED, __HIP_MEMORY_SCOPE_AGENT);
}

}

extern "C" void kernel_launch(int rank, int nranks, ncclComm_t comm,
                              hipStream_t stream,
                              void* const* d_in, const int* in_sizes, int n_in,
                              void* d_out, int out_size,
                              const int* in_dtypes, int out_dtype,
                              void* const* d_out_all,
                              void* const* d_ws_all, size_t ws_bytes) {
    (void)comm; (void)in_dtypes; (void)out_dtype; (void)d_out_all;
    if (nranks != NR || (unsigned)rank >= (unsigned)NR || n_in != 4 || ws_bytes < WS_NEED || in_sizes[0] != M * KL || in_sizes[1] != KL * N ||
        out_size != M * N) {
        fprintf(stderr, "kernel_launch: unexpected problem shape\n");
        return;
    }
    Params p;
    p.x = (const unsigned char*)d_in[0]; p.w = (const unsigned char*)d_in[1];
    p.sx = (const float*)d_in[2]; p.sw = (const float*)d_in[3];
    p.out = (float*)d_out; p.rank = rank;
    for (int i = 0; i < NR; ++i) p.ws[i] = (unsigned char*)d_ws_all[i];
    gemm_ar_kernel<<<NWG, 512, 0, stream>>>(p);
    HIP_CHECK(hipGetLastError());
}
```

```cpp
#include <hip/hip_runtime.h>
#include <rccl/rccl.h>
#include <cstdio>

#define HIP_CHECK(expr) do { hipError_t _e = (expr); if (_e != hipSuccess) { \
    fprintf(stderr, "HIP error %s at %s:%d\n", hipGetErrorString(_e), __FILE__, __LINE__); \
    return; } } while (0)

namespace {

#define LAS __attribute__((address_space(3)))
typedef int i32x4 __attribute__((ext_vector_type(4)));
typedef int i32x8 __attribute__((ext_vector_type(8)));
typedef float f32x4 __attribute__((ext_vector_type(4)));
typedef unsigned u32x2 __attribute__((ext_vector_type(2)));
typedef unsigned u32x4 __attribute__((ext_vector_type(4)));
typedef unsigned long long u64;

constexpr int NR = 8;
constexpr int M = 4096, N = 16384, KL = 512;
constexpr int NWG = 256, NPUSH = 16, NGEMM = NWG - NPUSH;
#define NXW_ 15
#define PW_ 2
#define WIN_ 21
#define CAUX_ 2
constexpr int NXW = NXW_;
constexpr int NBA = 16, NBB = 45;
constexpr int NBLK = NBA + NBB;
constexpr unsigned APIECE = 4096, BPIECE = 6144;
constexpr unsigned AKS = 8 * APIECE, BKS = 8 * BPIECE;
#define APAD_ 0
#define BPAD_ 0
constexpr unsigned ABLK = 32 * AKS + APAD_, BBLK = 32 * BKS + BPAD_;
constexpr unsigned BOFF = NBA * ABLK;
constexpr unsigned DATAB = BOFF + NBB * BBLK;
constexpr unsigned STG = AKS + BKS;
constexpr int NJOBS = NBLK * 32;
constexpr unsigned FLAGB = NBLK * 256;

constexpr size_t OFF_CNT = 0;
constexpr size_t OFF_TMO = 32768;
constexpr size_t OFF_TICKET = 33024;
constexpr size_t OFF_ROLE = 34816;
constexpr size_t OFF_FLAG = 65536;
constexpr size_t OFF_DATA = 1u << 20;
constexpr size_t WS_NEED = OFF_DATA + 2 * (size_t)DATAB;
constexpr u64 DEADLINE_TICKS = 50000000ull;
constexpr unsigned OOB = 0x80000000u;

struct Params {
    const unsigned char* x; const unsigned char* w; const float* sx; const float* sw;
    float* out; unsigned char* ws[NR]; int rank;
};

#define RSRC(ptr, bytes) __builtin_amdgcn_make_buffer_rsrc((void*)(ptr), 0, (unsigned)(bytes), 0x00020000)
#define WAIT_V(n) asm volatile("s_waitcnt vmcnt(" #n ")" ::: "memory")
#define WAIT_N(n) asm volatile("s_waitcnt vmcnt(%0)" :: "i"(n) : "memory")
#define WAIT_L(n) asm volatile("s_waitcnt lgkmcnt(" #n ")" ::: "memory")
#define BAR __builtin_amdgcn_s_barrier()
#define SCHED __builtin_amdgcn_sched_barrier(0)
constexpr int SYS = 17;
#define STAMP(i) do {} while (0)

constexpr int NXJ = NBA * 32, NWJ = NBB * 32;
__constant__ const unsigned WBLOCK[NBB] = {
    0x30000,0x31680,0x32b80,0x30180,0x31800,0x32d00,0x30300,0x31980,0x32e80,0x30480,0x31b00,0x33000,0x30600,0x31c80,0x33180,
    0x30780,0x31e00,0x33300,0x30900,0x31f80,0x33480,0x30a80,0x32100,0x33600,0x30c00,0x32280,0x23780,0x30d80,0x32400,0x23880,
    0x30f00,0x32580,0x23980,0x31080,0x32700,0x23a80,0x31200,0x22880,0x33b80,0x31380,0x22980,0x33d00,0x31500,0x22a80,0x33e80,
};
__constant__ const unsigned PUSH_ORDER[NWJ] = {
    0,1,2,384,385,3,386,96,97,4,387,98,480,481,5,388,99,482,192,193,6,389,100,483,
    194,576,577,7,390,101,484,195,578,288,289,8,9,391,102,485,196,579,290,672,673,10,392,393,
    103,486,197,580,291,674,11,394,104,105,487,198,581,292,675,12,395,106,488,489,199,582,293,676,
    13,396,107,490,200,201,583,294,677,14,397,108,491,202,584,585,295,678,15,398,109,492,203,586,
    296,297,679,16,17,399,110,493,204,587,298,680,681,18,400,401,111,494,205,588,299,682,19,402,
    112,113,495,206,589,300,683,20,403,114,496,497,207,590,301,684,21,404,115,768,769,498,208,209,
    591,302,685,1152,1153,22,405,116,770,499,864,865,210,592,593,303,686,1154,1248,1249,23,406,117,771,
    500,866,211,960,961,594,304,305,687,1155,1250,1344,1345,24,25,407,118,772,501,867,212,962,595,1056,
    1057,306,688,689,1156,1251,1346,26,408,409,119,773,502,868,213,963,596,1058,307,690,1157,1252,1347,27,
    410,120,121,774,503,869,214,964,597,1059,308,691,1158,1253,1348,28,411,122,775,504,505,870,215,965,
    598,1060,309,692,1159,1254,1349,29,412,123,776,777,506,871,216,217,966,599,1061,310,693,1160,1161,1255,
    1350,30,413,124,778,507,872,873,218,967,600,601,1062,311,694,1162,1256,1257,1351,31,414,125,779,508,
    874,219,968,969,602,1063,312,313,695,1163,1258,1352,1353,415,126,780,509,875,220,970,603,1064,1065,314,
    696,697,1164,1259,1354,127,781,510,876,221,971,604,1066,315,698,1165,1260,1355,782,511,877,222,972,605,
    1067,316,699,1166,1261,1356,783,878,223,973,606,1068,317,700,1167,1262,1357,784,785,879,974,607,1069,318,
    701,1168,1169,1263,1358,786,880,881,975,1070,319,702,1170,1264,1265,1359,32,33,787,882,976,977,1071,703,
    1171,1266,1360,1361,34,416,417,788,883,978,1072,1073,1172,1267,1362,35,418,128,129,789,884,979,1074,1173,
    1268,1363,36,419,130,512,513,790,885,980,1075,1174,1269,1364,37,420,131,514,791,224,225,886,981,1076,
    1175,1270,1365,38,421,132,515,792,793,226,887,608,609,982,1077,1176,1177,1271,1366,39,422,133,516,794,
    227,888,889,610,983,320,321,1078,1178,1272,1273,1367,40,41,423,134,517,795,228,890,611,984,985,322,
    1079,704,705,1179,1274,1368,1369,42,424,425,135,518,796,229,891,612,986,323,1080,1081,706,1180,1275,1370,
    43,426,136,137,519,797,230,892,613,987,324,1082,707,1181,1276,1371,44,427,138,520,521,798,231,893,
    614,988,325,1083,708,1182,1277,1372,45,428,139,522,799,232,233,894,615,989,326,1084,709,1183,1278,1373,
    46,429,140,523,234,895,616,617,990,327,1085,710,1279,1374,47,430,141,524,235,618,991,328,329,1086,
    711,1375,48,49,431,142,525,236,619,330,1087,712,713,50,432,433,143,526,237,620,331,714,51,434,
    144,145,527,238,621,332,715,52,435,146,528,529,239,622,333,716,53,436,147,1184,1185,800,801,530,
    240,241,623,334,717,1186,54,437,148,802,531,1280,1281,896,897,242,624,625,335,718,1187,1282,55,438,
    149,803,532,898,1188,243,1376,1377,992,993,626,336,337,719,1283,56,57,1189,1378,439,150,804,533,899,
    244,994,1284,627,1088,1089,338,720,721,1190,1379,58,440,441,1285,151,805,534,900,245,995,628,1090,1191,
    1380,339,722,1286,59,442,152,153,806,1192,1193,1381,535,901,246,996,629,1091,340,1287,723,1194,1382,60,
    443,154,807,536,537,902,1288,1289,247,997,630,1092,341,724,1195,1383,1290,61,444,155,808,809,538,903,
    248,249,998,1196,1384,1385,631,1093,342,725,1291,62,1197,1386,445,156,810,539,904,905,250,999,632,633,
    1094,1292,343,726,1198,1387,63,446,1293,157,811,540,906,251,1000,1001,634,1095,344,345,1199,1388,727,1294,
    447,158,812,1200,1201,1389,541,907,252,1002,635,1096,1097,346,728,729,1295,1202,1390,159,813,542,908,1296,
    1297,253,1003,636,1098,347,730,1203,1391,1298,814,543,909,254,1004,1204,1392,1393,637,1099,348,731,1299,1205,
    1394,815,910,255,1005,638,1100,1300,349,732,1206,1395,816,817,1301,911,1006,639,1101,350,1207,1396,733,1302,
    818,1208,1209,912,913,1397,1007,1102,351,734,1303,64,65,1210,1398,819,914,1304,1305,1008,1009,1103,735,1211,
    1399,66,448,449,1306,820,915,1010,1212,1400,1401,1104,1105,1307,67,450,160,161,1213,1402,821,916,1011,1106,
    1308,1214,1403,68,451,162,544,545,822,1309,917,1012,1107,1215,1404,69,1310,452,163,546,823,256,257,918,
    1405,1013,1108,1311,70,453,1406,164,547,824,825,258,919,640,641,1014,1109,1407,71,454,165,548,826,259,
    920,921,642,1015,352,353,1110,72,73,455,166,549,827,260,922,643,1016,1017,354,1111,736,737,74,456,
    457,167,550,828,261,923,644,1018,355,1112,1113,738,75,458,168,169,1216,1217,551,829,262,924,645,1019,
    356,1114,739,76,459,170,1218,552,553,1312,1313,830,263,925,646,1020,357,1115,740,77,460,171,1219,554,
    1314,831,264,265,1408,1409,926,647,1021,358,1116,741,78,461,172,1220,555,1315,266,1410,927,648,649,1022,
    359,1117,742,79,462,173,1221,556,1316,267,1411,650,1023,360,361,1118,743,80,81,463,174,1222,557,1317,
    268,1412,651,362,1119,744,745,82,464,465,175,1223,558,1318,269,1413,652,363,746,83,466,176,177,1224,
    1225,559,1319,270,1414,653,364,747,84,467,178,1226,560,561,1320,1321,271,1415,654,365,748,832,833,85,
    468,179,1227,562,1322,272,273,834,1416,1417,655,366,749,928,929,86,835,469,180,1228,563,1323,274,1418,
    656,657,930,367,750,836,1024,1025,87,470,931,181,1229,564,1324,275,1419,658,368,369,837,1026,751,932,
    1120,1121,88,89,471,182,1230,838,1027,565,1325,276,1420,659,370,752,753,933,1122,839,1028,90,472,473,
    183,1231,566,1326,934,1123,277,1421,660,371,754,840,841,1029,91,935,1124,474,184,185,1232,1233,567,1327,
    278,1422,842,1030,661,372,755,936,937,1125,92,475,843,1031,186,1234,568,569,1328,1329,279,1423,662,938,
    1126,373,756,844,1032,1033,93,476,187,939,1127,1235,570,1330,280,281,1424,1425,663,374,845,1034,757,940,
    1128,1129,94,477,188,1236,571,846,1035,1331,282,1426,664,665,375,758,941,1130,847,1036,95,478,189,1237,
    572,1332,283,942,1131,1427,666,376,377,759,848,849,1037,943,1132,479,190,1238,573,1333,284,1428,667,850,
    1038,378,760,761,944,945,1133,851,1039,191,1239,574,1334,285,1429,668,379,946,1134,762,852,1040,1041,1240,
    1241,947,1135,575,1335,286,1430,669,380,853,763,1042,948,1136,1137,1242,1336,1337,854,1043,287,1431,670,381,
    764,949,1138,855,1044,1243,1338,1432,1433,950,1139,671,382,765,856,857,1045,951,1140,1244,1339,1434,858,1046,
    383,766,952,953,1141,859,1047,1245,1340,1435,954,1142,767,860,1048,1049,1246,955,1143,1341,1436,861,1050,956,
    1144,1145,1247,1342,862,1051,1437,957,1146,863,1052,1343,1438,958,1147,1053,959,1148,1439,1054,1149,1055,1150,1151,
};
__device__ __forceinline__ void local_decode(int n, int& blk, int& c) {
    if (n < 992) { c = n / 31; const int u = n - c * 31; blk = u < NBA ? u : NBA + 3 * (u - NBA); }
    else { const int q = n - 992; c = q / 30; const int u = q - c * 30; blk = NBA + 3 * (u >> 1) + 1 + (u & 1); }
}
__device__ __forceinline__ unsigned piece_off(int blk, int pc) {
    return blk < NBA ? (unsigned)blk * ABLK + (unsigned)pc * APIECE : BOFF + (unsigned)(blk - NBA) * BBLK + (unsigned)pc * BPIECE;
}

struct XRegs { u32x4 q0, q1, q2, q3; };
struct WRegs { u32x4 q0, q1, q2, q3, q4, q5, q6, q7; };
struct WPack { u32x4 q0, q1, q2, q3, q4, q5; };
__device__ __forceinline__ XRegs x_load(const Params& p, int blk, int c, int lane) {
    const auto rs = RSRC(p.x, (unsigned)M * KL);
    const unsigned so = (unsigned)(blk * 256 * KL + c * 16), vo = (unsigned)lane * KL;
    XRegs R;
    R.q0 = __builtin_amdgcn_raw_buffer_load_b128(rs, vo, so, 0);
    R.q1 = __builtin_amdgcn_raw_buffer_load_b128(rs, vo, so + 64 * KL, 0);
    R.q2 = __builtin_amdgcn_raw_buffer_load_b128(rs, vo, so + 128 * KL, 0);
    R.q3 = __builtin_amdgcn_raw_buffer_load_b128(rs, vo, so + 192 * KL, 0);
    return R;
}
__device__ __forceinline__ WRegs w_load(const Params& p, unsigned desc, int c, int lane, bool live = true) {
    const auto rs = RSRC(p.w, (unsigned)KL * N);
    const unsigned so = live ? (unsigned)(c * 16 * N) + (desc & 0xffffu) : OOB, v1 = (unsigned)lane * 4u;
    const unsigned v2 = (desc >> 16) == 3 ? 256u + (v1 & 127u) : OOB / 2;
#define ROWS4(v, k) (u32x4){__builtin_amdgcn_raw_buffer_load_b32(rs, v, so + (k) * N, 0), __builtin_amdgcn_raw_buffer_load_b32(rs, v, so + ((k) + 1) * N, 0), \
                            __builtin_amdgcn_raw_buffer_load_b32(rs, v, so + ((k) + 2) * N, 0), __builtin_amdgcn_raw_buffer_load_b32(rs, v, so + ((k) + 3) * N, 0)}
    WRegs R;
    R.q0 = ROWS4(v1, 0); R.q1 = ROWS4(v1, 4); R.q2 = ROWS4(v1, 8); R.q3 = ROWS4(v1, 12);
    R.q4 = ROWS4(v2, 0); R.q5 = ROWS4(v2, 4); R.q6 = ROWS4(v2, 8); R.q7 = ROWS4(v2, 12);
#undef ROWS4
    return R;
}
__device__ __forceinline__ void tr4(const u32x4 r, int q, u32x4& c0, u32x4& c1, u32x4& c2, u32x4& c3) {
    const unsigned t0 = __builtin_amdgcn_perm(r[1], r[0], 0x05010400u), t1 = __builtin_amdgcn_perm(r[1], r[0], 0x07030602u);
    const unsigned t2 = __builtin_amdgcn_perm(r[3], r[2], 0x05010400u), t3 = __builtin_amdgcn_perm(r[3], r[2], 0x07030602u);
    c0[q] = __builtin_amdgcn_perm(t2, t0, 0x05040100u); c1[q] = __builtin_amdgcn_perm(t2, t0, 0x07060302u);
    c2[q] = __builtin_amdgcn_perm(t3, t1, 0x05040100u); c3[q] = __builtin_amdgcn_perm(t3, t1, 0x07060302u);
}
__device__ __forceinline__ WPack w_pack(int lane, const WRegs& R) {
    WPack P; u32x4 e0, e1, e2, e3;
    tr4(R.q0, 0, P.q0, P.q1, P.q2, P.q3); tr4(R.q1, 1, P.q0, P.q1, P.q2, P.q3); tr4(R.q2, 2, P.q0, P.q1, P.q2, P.q3); tr4(R.q3, 3, P.q0, P.q1, P.q2, P.q3);
    tr4(R.q4, 0, e0, e1, e2, e3); tr4(R.q5, 1, e0, e1, e2, e3); tr4(R.q6, 2, e0, e1, e2, e3); tr4(R.q7, 3, e0, e1, e2, e3);
    P.q4 = lane < 32 ? e0 : e1; P.q5 = lane < 32 ? e2 : e3;
    return P;
}
__device__ __forceinline__ unsigned char* uniform_ptr(unsigned char* q) {
    const u64 v = (u64)q;
    return (unsigned char*)(((u64)(unsigned)__builtin_amdgcn_readfirstlane((unsigned)(v >> 32)) << 32) | (unsigned)__builtin_amdgcn_readfirstlane((unsigned)v));
}

__device__ __forceinline__ bool is_pusher(int b) { return b < NPUSH; }
__device__ __forceinline__ int pusher_index(int b) { return b; }

__global__ void __launch_bounds__(256, 1) gemm_ar_kernel(Params p) {
    __shared__ __attribute__((aligned(16))) unsigned char smem[2 * STG];
    LAS unsigned char* lds = (LAS unsigned char*)smem;
    const int tid = threadIdx.x, wid = __builtin_amdgcn_readfirstlane(tid >> 6), lane = tid & 63;
    const int b = blockIdx.x;
    unsigned char* const me = p.ws[p.rank];
    unsigned* const cntp = (unsigned*)(me + OFF_CNT + (size_t)b * 128);
    const unsigned cnt = __hip_atomic_load(cntp, __ATOMIC_RELAXED, __HIP_MEMORY_SCOPE_AGENT);
#define CALL_STATE() const unsigned call = __builtin_amdgcn_readfirstlane(cnt) + 1u, par = call & 1u, tag = ((call + 1u) >> 1) & 0xffu
    const unsigned sA = (unsigned)lane * 16u;
    const unsigned sB0 = (unsigned)(lane >> 4) * 1024u + (unsigned)(lane & 15) * 16u;
    const unsigned sB1 = 4096u + (unsigned)((lane >> 4) & 1) * 1024u + (unsigned)(lane >> 5) * 256u + (unsigned)(lane & 15) * 16u;
    const unsigned sB2 = sB0 + (unsigned)(lane >> 5) * 1024u;

    if (is_pusher(b)) {
        if (wid >= PW_) return;
        constexpr int NW = NPUSH * PW_, NWW = NW - NXW;
        const int v = wid * NPUSH + pusher_index(b);
        int e = PUSH_ORDER[v < NXW ? 0 : v - NXW];
        unsigned desc = WBLOCK[e >> 5];
        XRegs R = x_load(p, v & 15, v >> 4, lane);
        WRegs RW = w_load(p, desc, e & 31, lane, v >= NXW);
        CALL_STATE();
        unsigned* const ticket = (unsigned*)(me + OFF_TICKET + par * 256);
        if (v == 0 && lane < 2) __hip_atomic_store((unsigned*)(me + OFF_TICKET + (par ^ 1u) * 256) + lane * 32, 0u, __ATOMIC_RELAXED, __HIP_MEMORY_SCOPE_AGENT);
        if (v == 0 && lane < 8) __hip_atomic_store((unsigned*)(me + OFF_ROLE + (par ^ 1u) * 1024) + lane * 32, 0u, __ATOMIC_RELAXED, __HIP_MEMORY_SCOPE_AGENT);
        unsigned char* dbase[NR - 1]; int rel[NR - 1];
#pragma unroll
        for (int q = 0; q < NR - 1; ++q) {
            const int peer = __builtin_amdgcn_readfirstlane((p.rank + 1 + (q + v) % (NR - 1)) & (NR - 1));
            rel[q] = __builtin_amdgcn_readfirstlane((p.rank - peer - 1) & (NR - 1));
            dbase[q] = uniform_ptr(p.ws[peer]) + OFF_DATA + (size_t)par * DATAB;
        }
        unsigned char* fptr;
        { const int q = lane < NR - 1 ? lane : 0, peer = (p.rank + 1 + q) & (NR - 1);
          fptr = p.ws[peer] + OFF_FLAG + (size_t)par * FLAGB + ((p.rank - peer - 1) & (NR - 1)); }
        int prev = -1, n0, n1;
#define PUSH7(d, vo, pz) do { _Pragma("unroll") for (int q = 0; q < NR - 1; ++q) \
            __builtin_amdgcn_raw_buffer_store_b128(d, RSRC(dbase[q], DATAB), vo, base + rel[q] * (pz), SYS); asm volatile("s_nop 1"); } while (0)
#define FLAG_PREV() do { if (prev >= 0 && lane < NR - 1) __hip_atomic_store(fptr + prev, (unsigned char)tag, __ATOMIC_RELAXED, __HIP_MEMORY_SCOPE_SYSTEM); } while (0)
#define DRAW(tk, which, cnt) do { tk = 0; if (lane == 0) asm volatile("global_atomic_add %0, %1, %2, %3 offset:%4 sc0" : "=v"(tk) : "v"(0u), "v"(cnt), "s"(ticket), "i"((which) * 128) : "memory"); } while (0)
        if (v < NXW) {
            n0 = v; n1 = v + NXW;
            asm volatile("" : "+v"(R.q0), "+v"(R.q1), "+v"(R.q2), "+v"(R.q3));
            while (n0 < NXJ) {
                const XRegs P = R;
                const int pc = (n0 & 15) * 256 + 32 + 7 * (n0 >> 4), nn = n1 < NXJ ? n1 : NXJ - 1;
                const unsigned base = piece_off(n0 & 15, pc & 255);
                unsigned tk;
                PUSH7(P.q0, sA, APIECE); WAIT_N(WIN_);
                R = x_load(p, nn & 15, nn >> 4, lane); DRAW(tk, 0, 1u);
                PUSH7(P.q1, sA + 1024u, APIECE); WAIT_N(19); FLAG_PREV();
                PUSH7(P.q2, sA + 2048u, APIECE); PUSH7(P.q3, sA + 3072u, APIECE); WAIT_N(WIN_);
                prev = pc;
                asm volatile("" : "+v"(tk), "+v"(R.q0), "+v"(R.q1), "+v"(R.q2), "+v"(R.q3));
                n0 = n1; n1 = 2 * NXW + (int)__builtin_amdgcn_readfirstlane(tk);
            }
            STAMP(2);
            unsigned tk; DRAW(tk, 1, 2u); WAIT_V(0); asm volatile("" : "+v"(tk));
            n0 = 2 * NWW + (int)__builtin_amdgcn_readfirstlane(tk); n1 = n0 + 1;
            e = PUSH_ORDER[n0 < NWJ ? n0 : 0]; desc = WBLOCK[e >> 5];
            RW = w_load(p, desc, e & 31, lane);
        } else { n0 = v - NXW; n1 = n0 + NWW; }
        if (n0 < NWJ) {
#define PINW(R) "+v"(R.q0), "+v"(R.q1), "+v"(R.q2), "+v"(R.q3), "+v"(R.q4), "+v"(R.q5), "+v"(R.q6), "+v"(R.q7)
            asm volatile("" : PINW(RW));
            while (n0 < NWJ) {
                const WPack P = w_pack(lane, RW);
                const int pc = (e >> 5) * 256 + 32 + 7 * (e & 31);
                const unsigned base = piece_off(NBA + (e >> 5), pc & 255), s0 = (desc >> 16) == 3 ? sB0 : sB2, s1 = (desc >> 16) == 3 ? sB1 : OOB;
                e = PUSH_ORDER[n1 < NWJ ? n1 : NWJ - 1]; desc = WBLOCK[e >> 5];
                unsigned tk;
                PUSH7(P.q0, s0, BPIECE); WAIT_N(WIN_);
                RW = w_load(p, desc, e & 31, lane); DRAW(tk, 1, 1u);
                PUSH7(P.q1, s0 + 256u, BPIECE); WAIT_N(47); FLAG_PREV();
                PUSH7(P.q2, s0 + 512u, BPIECE); PUSH7(P.q3, s0 + 768u, BPIECE); WAIT_N(WIN_);
                PUSH7(P.q4, s1, BPIECE); WAIT_N(WIN_); PUSH7(P.q5, s1 + 512u, BPIECE); WAIT_N(WIN_);
                prev = NBA * 256 + pc;
                asm volatile("" : "+v"(tk), PINW(RW));
                n0 = n1; n1 = 2 * NWW + (int)__builtin_amdgcn_readfirstlane(tk);
            }
        }
        WAIT_V(0);
        FLAG_PREV();
        if (tid == 0) __hip_atomic_store(cntp, call, __ATOMIC_RELAXED, __HIP_MEMORY_SCOPE_AGENT);
        STAMP(1);
        return;
    }

    CALL_STATE();
    if (tid == 0) {
        const unsigned xcc = (unsigned)__builtin_amdgcn_s_getreg((3 << 11) | 20) & 7u;
        int role = 0;
        for (unsigned k = 0; k < 8; ++k) {
            const unsigned x = (xcc + 2 * k + (k >> 2)) & 7u;
            const unsigned t = __hip_atomic_fetch_add((unsigned*)(me + OFF_ROLE + par * 1024) + x * 32, 1u, __ATOMIC_RELAXED, __HIP_MEMORY_SCOPE_AGENT);
            if (t < 30u) { role = (int)(((x & 4u) + ((x >> 1) & 1u) + ((x & 1u) ? 0u : 2u)) * 30u + t); break; }
        }
        *(LAS int*)lds = role;
    }
    __syncthreads();
    const int gw = __builtin_amdgcn_readfirstlane(*(LAS int*)lds);
    __syncthreads();
    const auto rsD = RSRC(me + OFF_DATA + (size_t)par * DATAB, DATAB);
    const auto rsC = RSRC(me, OFF_DATA);
    const unsigned fbase = (unsigned)OFF_FLAG + par * FLAGB;
    {
        constexpr int NW = NGEMM * 4;
        const int v = gw + NGEMM * wid;
#define PUT(d, vo) do { __builtin_amdgcn_raw_buffer_store_b128(d, rsD, vo, base, SYS); asm volatile("s_nop 1"); } while (0)
        for (int n = v; n < NJOBS; n += NW) {
            int blk, c; local_decode(n, blk, c);
            const unsigned base = piece_off(blk, c);
            if (blk < NBA) {
                const XRegs P = x_load(p, blk, c, lane);
                PUT(P.q0, sA); PUT(P.q1, sA + 1024u); PUT(P.q2, sA + 2048u); PUT(P.q3, sA + 3072u);
            } else {
                const unsigned desc = WBLOCK[blk - NBA];
                const WPack P = w_pack(lane, w_load(p, desc, c, lane));
                const unsigned s0 = (desc >> 16) == 3 ? sB0 : sB2, s1 = (desc >> 16) == 3 ? sB1 : OOB;
                PUT(P.q0, s0); PUT(P.q1, s0 + 256u); PUT(P.q2, s0 + 512u); PUT(P.q3, s0 + 768u); PUT(P.q4, s1); PUT(P.q5, s1 + 512u);
            }
            WAIT_V(0);
            if (lane == 0) __builtin_amdgcn_raw_buffer_store_b8((unsigned char)tag, rsC, (unsigned)(blk * 256 + c), fbase, SYS);
        }
    }
    STAMP(1);

    const int wr = wid >> 1, wc = wid & 1;
    const unsigned scale_bits = __builtin_amdgcn_readfirstlane(__builtin_bit_cast(unsigned, p.sx[0] * p.sw[0]));
    const unsigned tag4 = tag * 0x01010101u;
    const unsigned vA = wid * 2 * APIECE + lane * 16, vB = wid * 2 * BPIECE + lane * 16;
    const unsigned lA = wid * 2 * APIECE, lB = AKS + wid * 2 * BPIECE;
    unsigned a0 = (unsigned)(size_t)lds + (lane >> 4) * APIECE + (wr * 128 + (lane & 15)) * 16;
    unsigned b0 = (unsigned)(size_t)lds + AKS + (lane >> 4) * BPIECE + (wc * 192 + (lane & 15)) * 16;
    unsigned a1 = a0 + STG, b1 = b0 + STG;
    asm volatile("" : "+v"(a0), "+v"(b0), "+v"(a1), "+v"(b1));
#define DMA(v, ldsoff, soff, imm) __builtin_amdgcn_raw_ptr_buffer_load_lds(rsD, (LAS void*)(lds + (ldsoff)), 16, v, soff, imm, SYS)
#define DMA_A(st, so, q) DMA(vA, (st) * STG + lA + ((q) >> 2) * 4096, (so) + ((q) >> 2) * 4096, ((q) & 3) * 1024)
#define DMA_B(st, so, s, g6) DMA(vB, (st) * STG + lB + (s) * BPIECE + ((g6) >> 2) * 4096, (so) + (s) * BPIECE + ((g6) >> 2) * 4096, ((g6) & 3) * 1024)
#define DSR(dst, base, off) asm volatile("ds_read_b128 %0, %1 offset:%2" : "=v"(dst) : "v"(base), "i"(off) : "memory")
#define LDA(m, ab) do { i32x4 lo_, hi_; DSR(lo_, ab, (m) * 256); DSR(hi_, ab, (m) * 256 + 4 * APIECE); \
        A[m] = __builtin_shufflevector(lo_, hi_, 0, 1, 2, 3, 4, 5, 6, 7); } while (0)
#define LDB(dst, j, bb) do { i32x4 lo_, hi_; DSR(lo_, bb, ((j) >> 2) * 1024 + ((j) & 3) * 256); DSR(hi_, bb, ((j) >> 2) * 1024 + ((j) & 3) * 256 + 4 * BPIECE); \
        dst = __builtin_shufflevector(lo_, hi_, 0, 1, 2, 3, 4, 5, 6, 7); } while (0)
#define MF(m, j) do { if ((j) < 8) asm volatile("v_mfma_f32_16x16x128_f8f6f4 %0, %1, %2, %0" : "+a"(acc[j][m]) : "v"(A[m]), "v"(Bq[(j) & 1])); \
                      else asm volatile("v_mfma_f32_16x16x128_f8f6f4 %0, %1, %2, %0" : "+v"(acc[j][m]) : "v"(A[m]), "v"(Bq[(j) & 1])); } while (0)

    u64 sm = 0, nm = 0;
    unsigned sf, nf;
    u32x2 fl = {0u, 0u};
    int fl_tgt = 0;
    unsigned tv = 0;
#define FLAG_OFF(mb_, nb_) ((unsigned)((lane < 32 ? (mb_) : NBA + (nb_)) * 256 + (lane & 31) * 8))
#define READY(f) __builtin_amdgcn_ballot_w64((f)[0] == tag4 && (f)[1] == tag4)
#define NOW(t) do { u64 t64_; asm volatile("s_memrealtime %0\n\ts_waitcnt lgkmcnt(0)" : "=s"(t64_) :: "memory"); t = (unsigned)t64_; } while (0)
#define NEED(ks) do { const u64 need_ = 0x100000001ull << (ks); \
        if ((sm & need_) != need_) { \
            unsigned t0_, t1_; NOW(t0_); \
            for (;;) { u32x2 f_; unsigned t_; \
                asm volatile("buffer_load_dwordx2 %0, %2, %3, %4 offen sc0 sc1\n\tbuffer_load_dword %1, off, %3, %5 sc1\n\ts_waitcnt vmcnt(0)" \
                             : "=&v"(f_), "=&v"(t_) : "v"(sf), "s"(rsC), "s"(fbase), "s"((unsigned)OFF_TMO) : "memory"); \
                sm = READY(f_); \
                if ((sm & need_) == need_) break; \
                NOW(t1_); \
                if (__builtin_amdgcn_readfirstlane(t_) == call || t1_ - t0_ > (unsigned)DEADLINE_TICKS) { \
                    unsigned c_ = call; asm volatile("" : "+v"(c_)); \
                    __builtin_amdgcn_raw_buffer_store_b32(c_, rsC, 0, (unsigned)OFF_TMO, 16); asm volatile("s_nop 1"); \
                    sm = ~0ull; break; } \
                __builtin_amdgcn_s_sleep(1); } \
        } } while (0)
#define REFRESH() do { asm volatile("" : "+v"(fl)); \
        const u64 m_ = READY(fl); sm |= fl_tgt == 1 ? m_ : 0ull; nm |= fl_tgt == 2 ? m_ : 0ull; \
        fl_tgt = sm != ~0ull ? 1 : 2; \
        const unsigned fo_ = fl_tgt == 1 ? sf : nf; \
        asm volatile("buffer_load_dwordx2 %0, %2, %3, %4 offen sc0 sc1\n\tbuffer_load_dword %1, off, %3, %5 sc1" \
                     : "+v"(fl), "+v"(tv) : "v"(fo_), "s"(rsC), "s"(fbase), "s"((unsigned)OFF_TMO) : "memory"); } while (0)

#define COL(j, bc, F1, F2, F3) do { \
        WAIT_L(0); MF(0, j); LDB(Bq[((j) + 1) & 1], (j) + 1, bc); \
        MF(1, j); F1; SCHED; MF(2, j); MF(3, j); F2; SCHED; MF(4, j); MF(5, j); F3; SCHED; MF(6, j); MF(7, j); SCHED; } while (0)
#define COL0(bc, F1, F2) do { \
        WAIT_L(14); MF(0, 0); LDB(Bq[1], 1, bc); WAIT_L(14); MF(1, 0); F1; SCHED; WAIT_L(12); MF(2, 0); WAIT_L(10); MF(3, 0); WAIT_L(8); MF(4, 0); F2; SCHED; \
        WAIT_L(6); MF(5, 0); WAIT_L(4); MF(6, 0); WAIT_L(2); MF(7, 0); SCHED; } while (0)
#define COLL(j, bn, an, F1, F2) do { \
        MF(0, j); LDB(Bq[0], 0, bn); LDA(0, an); MF(1, j); LDA(1, an); F1; SCHED; MF(2, j); LDA(2, an); MF(3, j); LDA(3, an); \
        MF(4, j); LDA(4, an); F2; SCHED; MF(5, j); LDA(5, an); MF(6, j); LDA(6, an); MF(7, j); LDA(7, an); SCHED; } while (0)
#define SYNC() do { asm volatile("s_waitcnt vmcnt(20) lgkmcnt(0)" ::: "memory"); BAR; SCHED; } while (0)
#define KHEAD(st, bc, bp, bpx, WV) do { \
        COL0(bc, DMA_B((st) ^ 1, bp, 1, 1), DMA_B((st) ^ 1, bp, 1, 4)); \
        COL(1, bc, DMA_B((st) ^ 1, bpx, 0, 2), , DMA_B((st) ^ 1, bpx, 0, 5)); \
        COL(2, bc, DMA_B((st) ^ 1, bpx, 1, 2), , DMA_B((st) ^ 1, bpx, 1, 5)); \
        asm volatile("s_waitcnt vmcnt(" #WV ") lgkmcnt(0)" : "+v"(tv) :: "memory"); BAR; SCHED; } while (0)
#define EARLY(a2, b2, k2) const bool rdy_ = ((sm >> (k2)) & 0x100000001ull) == 0x100000001ull; \
        const unsigned ae_ = rdy_ ? (a2) : OOB, be_ = rdy_ ? (b2) : OOB; SCHED
#define LATE(st, a2, b2, k2) do { if (!rdy_) { NEED(k2); \
        DMA_A(st, a2, 0); DMA_A(st, a2, 1); DMA_A(st, a2, 2); DMA_A(st, a2, 3); DMA_A(st, a2, 4); DMA_A(st, a2, 5); DMA_A(st, a2, 6); DMA_A(st, a2, 7); \
        DMA_B(st, b2, 0, 0); DMA_B(st, b2, 0, 3); DMA_B(st, b2, 1, 0); DMA_B(st, b2, 1, 3); } } while (0)
#define KSTEP3(st, bc, an, bn, bp, bpx, a2, b2, k2, RF) do { \
        KHEAD(st, bc, bp, bpx, 20); EARLY(a2, b2, k2); \
        COL(3, bc, DMA_A(st, ae_, 0), , DMA_A(st, ae_, 1)); \
        COL(4, bc, DMA_A(st, ae_, 2), , DMA_A(st, ae_, 3)); \
        COL(5, bc, DMA_A(st, ae_, 4), , DMA_A(st, ae_, 5)); \
        COL(6, bc, DMA_A(st, ae_, 6), , DMA_A(st, ae_, 7)); \
        COL(7, bc, DMA_B(st, be_, 0, 0), , RF); \
        COL(8, bc, DMA_B(st, be_, 0, 3), , ); \
        COL(9, bc, DMA_B(st, be_, 1, 0), , ); \
        COL(10, bc, DMA_B(st, be_, 1, 3), , ); \
        SYNC(); LATE(st, a2, b2, k2); SCHED; \
        COLL(11, bn, an, DMA_B(st, b2, 0, 1), DMA_B(st, b2, 0, 4)); } while (0)
#define KSTEP2(st, bc, an, bn, bp, bpx, a2, b2, k2, RF) do { \
        KHEAD(st, bc, bp, bpx, 20); EARLY(a2, b2, k2); \
        COL(3, bc, DMA_A(st, ae_, 0), DMA_A(st, ae_, 1), DMA_A(st, ae_, 2)); \
        COL(4, bc, DMA_A(st, ae_, 3), DMA_A(st, ae_, 4), DMA_A(st, ae_, 5)); \
        COL(5, bc, DMA_A(st, ae_, 6), DMA_A(st, ae_, 7); RF, DMA_B(st, be_, 0, 0)); \
        COL(6, bc, DMA_B(st, be_, 0, 3), DMA_B(st, be_, 1, 0), DMA_B(st, be_, 1, 3)); \
        SYNC(); LATE(st, a2, b2, k2); SCHED; \
        COLL(7, bn, an, DMA_B(st, b2, 0, 1), DMA_B(st, b2, 0, 4)); } while (0)
#define EQ(g, m, t) do { float x0, x1, x2, x3; \
        if ((g) < 2) asm volatile("v_accvgpr_read_b32 %0, %4\n\tv_accvgpr_read_b32 %1, %5\n\tv_accvgpr_read_b32 %2, %6\n\tv_accvgpr_read_b32 %3, %7\n\t" \
                                  "v_mul_f32 %0, %8, %0\n\tv_mul_f32 %1, %8, %1\n\tv_mul_f32 %2, %8, %2\n\tv_mul_f32 %3, %8, %3" \
                                  : "=&v"(x0), "=&v"(x1), "=&v"(x2), "=&v"(x3) \
                                  : "a"(acc[4 * (g)][m][t]), "a"(acc[4 * (g) + 1][m][t]), "a"(acc[4 * (g) + 2][m][t]), "a"(acc[4 * (g) + 3][m][t]), "s"(sc)); \
        else asm volatile("v_mul_f32 %0, %8, %4\n\tv_mul_f32 %1, %8, %5\n\tv_mul_f32 %2, %8, %6\n\tv_mul_f32 %3, %8, %7" \
                          : "=&v"(x0), "=&v"(x1), "=&v"(x2), "=&v"(x3) \
                          : "v"(acc[4 * (g)][m][t]), "v"(acc[4 * (g) + 1][m][t]), "v"(acc[4 * (g) + 2][m][t]), "v"(acc[4 * (g) + 3][m][t]), "s"(sc)); \
        const f32x4 v_ = {x0, x1, x2, x3}; \
        __builtin_amdgcn_raw_buffer_store_b128(__builtin_bit_cast(u32x4, v_), rs, o + (g) * 256u, 0, CAUX_); \
        o += ((t) == 3 ? ((m) == 7 ? 0u - 115u : 13u) : 1u) * N * 4u; asm volatile("" : "+v"(o)); SCHED; } while (0)
#define COLE(j, bc, g, q, F1, F2, F3) do { \
        WAIT_L(0); MF(0, j); LDB(Bq[((j) + 1) & 1], (j) + 1, bc); EQ(g, 2 * (q), 0); MF(1, j); F1; EQ(g, 2 * (q), 1); MF(2, j); EQ(g, 2 * (q), 2); \
        MF(3, j); F2; EQ(g, 2 * (q), 3); MF(4, j); EQ(g, 2 * (q) + 1, 0); MF(5, j); F3; EQ(g, 2 * (q) + 1, 1); MF(6, j); EQ(g, 2 * (q) + 1, 2); \
        MF(7, j); EQ(g, 2 * (q) + 1, 3); } while (0)
#define COLLE(j, g, F1, F2) do { \
        MF(0, j); EQ(g, 6, 0); MF(1, j); F1; EQ(g, 6, 1); MF(2, j); EQ(g, 6, 2); MF(3, j); EQ(g, 6, 3); \
        MF(4, j); F2; EQ(g, 7, 0); MF(5, j); EQ(g, 7, 1); MF(6, j); EQ(g, 7, 2); MF(7, j); EQ(g, 7, 3); } while (0)
#define EPI_SETUP(NG) \
        const unsigned sc = __builtin_amdgcn_readfirstlane(tv) == call ? 0x7fc00000u : scale_bits; \
        const auto rs = RSRC(p.out + (size_t)mb * 256 * N + (desc & 0xffffu), 256u * N * 4u - (desc & 0xffffu) * 4u); \
        const unsigned ln = __builtin_amdgcn_mbcnt_hi(~0u, __builtin_amdgcn_mbcnt_lo(~0u, 0u)); \
        unsigned o = (unsigned)((wr * 128 + 4 * (ln >> 4)) * N + wc * 64 * NG + 4 * (ln & 15)) * 4u; SCHED
#define SYNC_L() do { asm volatile("s_waitcnt lgkmcnt(0)" ::: "memory"); BAR; SCHED; } while (0)
#define KLAST3(bc, bp, bpx, a2, b2, k2) \
        KHEAD(1, bc, bp, bpx, 8); EARLY(a2, b2, k2); EPI_SETUP(3); \
        COL(3, bc, DMA_A(1, ae_, 0), , DMA_A(1, ae_, 1)); \
        COLE(4, bc, 0, 0, DMA_A(1, ae_, 2), , DMA_A(1, ae_, 3)); \
        COLE(5, bc, 0, 1, DMA_A(1, ae_, 4), , DMA_A(1, ae_, 5)); \
        COLE(6, bc, 0, 2, DMA_A(1, ae_, 6), , DMA_A(1, ae_, 7)); \
        COLE(7, bc, 0, 3, DMA_B(1, be_, 0, 0), , ); \
        COLE(8, bc, 1, 0, DMA_B(1, be_, 0, 3), , ); \
        COLE(9, bc, 1, 1, DMA_B(1, be_, 1, 0), , ); \
        COLE(10, bc, 1, 2, DMA_B(1, be_, 1, 3), , ); \
        SYNC_L(); LATE(1, a2, b2, k2); SCHED; \
        COLLE(11, 1, DMA_B(1, b2, 0, 1), DMA_B(1, b2, 0, 4))
#define KLAST2(bc, bp, bpx, a2, b2, k2) \
        KHEAD(1, bc, bp, bpx, 8); EARLY(a2, b2, k2); EPI_SETUP(2); \
        COL(3, bc, DMA_A(1, ae_, 0), DMA_A(1, ae_, 1), DMA_A(1, ae_, 2)); \
        COLE(4, bc, 0, 0, DMA_A(1, ae_, 3), DMA_A(1, ae_, 4), DMA_A(1, ae_, 5)); \
        COLE(5, bc, 0, 1, DMA_A(1, ae_, 6), DMA_A(1, ae_, 7), DMA_B(1, be_, 0, 0)); \
        COLE(6, bc, 0, 2, DMA_B(1, be_, 0, 3), DMA_B(1, be_, 1, 0), DMA_B(1, be_, 1, 3)); \
        SYNC_L(); LATE(1, a2, b2, k2); SCHED; \
        COLLE(7, 0, DMA_B(1, b2, 0, 1), DMA_B(1, b2, 0, 4))
#define TILE(NG, KS, KL) { \
        f32x4 acc[4 * NG][8]; \
        i32x8 A[8], Bq[2]; \
        LDB(Bq[0], 0, b0); LDA(0, a0); LDA(1, a0); LDA(2, a0); LDA(3, a0); LDA(4, a0); LDA(5, a0); LDA(6, a0); LDA(7, a0); \
        u64 z_ = 0; asm volatile("s_nop 1" : "+v"(z_)); \
        _Pragma("unroll") for (int j = 0; j < 4 * NG; ++j) _Pragma("unroll") for (int m = 0; m < 8; ++m) { \
            if (j < 8) asm volatile("v_mfma_f32_4x4x4_16b_f16 %0, %1, %1, 0" : "=a"(acc[j][m]) : "v"(z_)); \
            else asm volatile("v_mfma_f32_4x4x4_16b_f16 %0, %1, %1, 0" : "=v"(acc[j][m]) : "v"(z_)); } \
        asm volatile("s_nop 7"); SCHED; \
        const unsigned nBx = nng == 3 ? nB : OOB; \
        for (int t = 0;; t += 2) { \
            const bool last = (t == 30); \
            const unsigned a2 = cA + (unsigned)(last ? 0 : t + 2) * AKS, b2 = last ? nB : cB + (unsigned)(t + 2) * BKS; \
            const unsigned b2x = NG == 3 ? b2 : OOB; \
            const int k2 = last ? 0 : t + 2; \
            if (last) { sm = nm; sf = nf; fl_tgt = fl_tgt == 2 ? 1 : 0; } \
            KS(0, b0, a1, b1, bprev, bprevx, a2, b2, k2, REFRESH()); \
            if (last) break; \
            KS(1, b1, a0, b0, b2, b2x, a2 + AKS, b2 + BKS, k2 + 1, ); \
            bprev = b2 + BKS; bprevx = b2x + BKS; \
            KSTAMP(); \
        } \
        KL(b1, nB, nBx, cA + AKS, nB + BKS, 1); \
        bprev = nB + BKS; bprevx = nBx + BKS; \
        asm volatile("s_nop 15\n\ts_nop 7" ::: "memory"); SCHED; \
        STAMP(12 + r); \
        _Pragma("unroll") for (int m = 0; m < 8; ++m) _Pragma("unroll") for (int t = 0; t < 4; ++t) EQ(NG - 1, m, t); \
    }
#define KSTAMP() do {} while (0)

    const int mb = 8 * (gw / 120) + (gw & 7), wl = 3 * ((gw % 120) >> 3);
    const unsigned cA = (unsigned)mb * ABLK;
    unsigned cB = BOFF + (unsigned)wl * BBLK;
    unsigned desc = WBLOCK[wl];
    sf = FLAG_OFF(mb, wl);
    NEED(0); NEED(1);
    {
        const unsigned cBx = (desc >> 16) == 3 ? cB : OOB;
        DMA_A(0, cA, 0); DMA_A(0, cA, 1); DMA_A(0, cA, 2); DMA_A(0, cA, 3); DMA_A(0, cA, 4); DMA_A(0, cA, 5); DMA_A(0, cA, 6); DMA_A(0, cA, 7);
        DMA_B(0, cB, 0, 0); DMA_B(0, cB, 0, 3); DMA_B(0, cB, 1, 0); DMA_B(0, cB, 1, 3);
        DMA_B(0, cB, 0, 1); DMA_B(0, cB, 0, 4); DMA_B(0, cB, 1, 1); DMA_B(0, cB, 1, 4); DMA_B(0, cBx, 0, 2); DMA_B(0, cBx, 0, 5); DMA_B(0, cBx, 1, 2); DMA_B(0, cBx, 1, 5);
        DMA_A(1, cA + AKS, 0); DMA_A(1, cA + AKS, 1); DMA_A(1, cA + AKS, 2); DMA_A(1, cA + AKS, 3);
        DMA_A(1, cA + AKS, 4); DMA_A(1, cA + AKS, 5); DMA_A(1, cA + AKS, 6); DMA_A(1, cA + AKS, 7);
        DMA_B(1, cB + BKS, 0, 0); DMA_B(1, cB + BKS, 0, 3); DMA_B(1, cB + BKS, 1, 0); DMA_B(1, cB + BKS, 1, 3);
    }
    SYNC();
    DMA_B(1, cB + BKS, 0, 1); DMA_B(1, cB + BKS, 0, 4);
    unsigned bprev = cB + BKS, bprevx = (desc >> 16) == 3 ? bprev : OOB;
    STAMP(2);
    for (int r = 0; r < 3; ++r) {
        const int nr = r < 2 ? r + 1 : r;
        const unsigned ndesc = WBLOCK[wl + nr], nB = BOFF + (unsigned)(wl + nr) * BBLK;
        const unsigned ng = desc >> 16, nng = ndesc >> 16;
        nf = FLAG_OFF(mb, wl + nr); nm = 0;
        if (fl_tgt == 2) fl_tgt = 0;
        SCHED;
        if (ng == 3) { TILE(3, KSTEP3, KLAST3) } else { TILE(2, KSTEP2, KLAST2) }
        STAMP(3 + r);
        desc = ndesc; cB = nB;
    }
    asm volatile("s_waitcnt vmcnt(0) lgkmcnt(0)" ::: "memory");
    STAMP(15);
    if (tid == 0) __hip_atomic_store(cntp, call, __ATOMIC_RELAXED, __HIP_MEMORY_SCOPE_AGENT);
}

}

extern "C" void kernel_launch(int rank, int nranks, ncclComm_t comm,
                              hipStream_t stream,
                              void* const* d_in, const int* in_sizes, int n_in,
                              void* d_out, int out_size,
                              const int* in_dtypes, int out_dtype,
                              void* const* d_out_all,
                              void* const* d_ws_all, size_t ws_bytes) {
    (void)comm; (void)in_dtypes; (void)out_dtype; (void)d_out_all;
    if (nranks != NR || (unsigned)rank >= (unsigned)NR || n_in != 4 || ws_bytes < WS_NEED || in_sizes[0] != M * KL || in_sizes[1] != KL * N ||
        out_size != M * N) {
        fprintf(stderr, "kernel_launch: unexpected problem shape\n");
        return;
    }
    Params p;
    p.x = (const unsigned char*)d_in[0]; p.w = (const unsigned char*)d_in[1];
    p.sx = (const float*)d_in[2]; p.sw = (const float*)d_in[3];
    p.out = (float*)d_out; p.rank = rank;
    for (int i = 0; i < NR; ++i) p.ws[i] = (unsigned char*)d_ws_all[i];
    gemm_ar_kernel<<<NWG, 256, 0, stream>>>(p);
    HIP_CHECK(hipGetLastError());
}
```
